```python
import numpy as np

kernel_name = "gpu_ag_gemm_m4096_k4096_n16384_w8a8_dequant"


ABSMAX_RTOL = 0.001

COMPUTE_DTYPE = "int8"
EPILOGUE = "dequant"
IN_QUANT = "int8"
OUT_QUANT = None
IN_DTYPES = {"x": "int8", "w_mat": "int8",
             "scale_x": "float32", "scale_w": "float32"}
WORLD_SIZE = 8

def setup_inputs():

    x = np.random.randint(-127, 128, size=(4096, 4096)).astype(np.int8)
    w_mat = np.random.randint(-127, 128, size=(4096, 16384)).astype(np.int8)
    scale_x = (0.02 * (1.0 + np.random.rand(1))).astype(np.float32)
    scale_w = (0.02 * (1.0 + np.random.rand(1))).astype(np.float32)
    return {"x": x, "w_mat": w_mat, "scale_x": scale_x, "scale_w": scale_w}

def reference(x, w_mat, scale_x, scale_w):
    acc = x.astype(np.float64) @ w_mat.astype(np.float64)
    y = (acc.astype(np.float32) * (scale_x * scale_w)).astype(np.float32)
    return y
```

```cpp
#include <hip/hip_runtime.h>
#include <rccl/rccl.h>
#include <cstdio>

#define HIP_CHECK(expr) do { hipError_t _e = (expr); if (_e != hipSuccess) { \
    fprintf(stderr, "HIP error %s at %s:%d\n", hipGetErrorString(_e), __FILE__, __LINE__); \
    return; } } while (0)

typedef unsigned u32x4 __attribute__((ext_vector_type(4)));
#define GAS __attribute__((address_space(1)))

constexpr int NR = 8;
constexpr int KD = 4096;
constexpr int NL = 2048;
constexpr int MS = 512;
constexpr int NITEM = MS / 32;
constexpr unsigned SHARD = MS * KD;
constexpr unsigned QUART = SHARD / 4;
constexpr int NPUSH = 32, GRID = 256, THREADS = 256;
constexpr unsigned WS_CNT = 0;
constexpr unsigned WS_TMO = GRID * 4 * 128;
constexpr unsigned WS_RDY = WS_TMO + 128;
constexpr unsigned WS_PROG = WS_RDY + 128;
constexpr unsigned WS_DATA = 1u << 20;
constexpr size_t WS_NEED = (size_t)WS_DATA + 2ull * NR * SHARD;
constexpr unsigned DEADLINE_TICKS = 50000000u;
constexpr int LDS_SCALE = 4 * 32768;
constexpr int LDS_BYTES = LDS_SCALE + 16;

struct Params {
    const char* x; const char* w; const float* sx; const float* sw; float* out;
    char* ws[NR];
    int rank; int pad;
};

template <int N> __device__ __forceinline__ void wait_vm() { asm volatile("s_waitcnt vmcnt(%0)" ::"i"(N) : "memory"); }
__device__ __forceinline__ const char* sptr(const char* p) {
    unsigned long long v = (unsigned long long)p;
    unsigned lo = __builtin_amdgcn_readfirstlane((unsigned)v), hi = __builtin_amdgcn_readfirstlane((unsigned)(v >> 32));
    unsigned long long r = ((unsigned long long)hi << 32) | lo;
    asm volatile("s_nop 4" : "+s"(r));
    return (const char*)r;
}

struct Chunk { u32x4 v[8]; };
__device__ __forceinline__ void push_load(Chunk& c, const char* sb, unsigned vo) {
    asm volatile("global_load_dwordx4 %0, %8, %9\n\t"
                 "global_load_dwordx4 %1, %8, %9 offset:64\n\t"
                 "global_load_dwordx4 %2, %8, %9 offset:128\n\t"
                 "global_load_dwordx4 %3, %8, %9 offset:192\n\t"
                 "global_load_dwordx4 %4, %8, %9 offset:256\n\t"
                 "global_load_dwordx4 %5, %8, %9 offset:320\n\t"
                 "global_load_dwordx4 %6, %8, %9 offset:384\n\t"
                 "global_load_dwordx4 %7, %8, %9 offset:448"
                 : "=&v"(c.v[0]), "=&v"(c.v[1]), "=&v"(c.v[2]), "=&v"(c.v[3]), "=&v"(c.v[4]), "=&v"(c.v[5]), "=&v"(c.v[6]), "=&v"(c.v[7])
                 : "v"(vo), "s"(sb) : "memory");
}
template <int N> __device__ __forceinline__ void push_wait(Chunk& c) {
    asm volatile("s_waitcnt vmcnt(%8)"
                 : "+v"(c.v[0]), "+v"(c.v[1]), "+v"(c.v[2]), "+v"(c.v[3]), "+v"(c.v[4]), "+v"(c.v[5]), "+v"(c.v[6]), "+v"(c.v[7])
                 : "i"(N) : "memory");
}
__device__ __forceinline__ void push_store(const Chunk& c, const char* sb  , unsigned vo) {
    asm volatile("global_store_dwordx4 %8, %0, %9 offset:-4096 sc0 sc1\n\t"
                 "global_store_dwordx4 %8, %1, %9 offset:-3072 sc0 sc1\n\t"
                 "global_store_dwordx4 %8, %2, %9 offset:-2048 sc0 sc1\n\t"
                 "global_store_dwordx4 %8, %3, %9 offset:-1024 sc0 sc1\n\t"
                 "global_store_dwordx4 %8, %4, %9 sc0 sc1\n\t"
                 "global_store_dwordx4 %8, %5, %9 offset:1024 sc0 sc1\n\t"
                 "global_store_dwordx4 %8, %6, %9 offset:2048 sc0 sc1\n\t"
                 "global_store_dwordx4 %8, %7, %9 offset:3072 sc0 sc1\n\t"
                 "s_nop 1"
                 :: "v"(c.v[0]), "v"(c.v[1]), "v"(c.v[2]), "v"(c.v[3]), "v"(c.v[4]), "v"(c.v[5]), "v"(c.v[6]), "v"(c.v[7]),
                    "v"(vo), "s"(sb) : "memory");
}
__device__ __forceinline__ void push_flag(const char* flag, unsigned zero, unsigned val) {
    asm volatile("s_mov_b64 exec, 1\n\tglobal_store_dword %0, %1, %2 sc0 sc1\n\ts_mov_b64 exec, -1" ::"v"(zero), "v"(val), "s"(flag) : "memory");
}

__device__ __forceinline__ void pusher(const Params& p, int lane) {
    const int label = blockIdx.x & 7, kq = blockIdx.x >> 3;
    char* myws = p.ws[0];
    char* pws = p.ws[label];
    constexpr unsigned IT = 32u * KD, H = 16u * KD;
    const unsigned svo = (unsigned)(lane & 15) * KD + (unsigned)(lane >> 4) * 16u + (unsigned)kq * 1024u;
    const char* xs = sptr(p.x);

    unsigned ticket, call;
    asm volatile("s_mov_b64 exec, 1\n\tglobal_atomic_add %0, %1, %2, %3 sc0\n\ts_mov_b64 exec, -1"
                 : "=&v"(ticket) : "v"(0u), "v"(1u), "s"(sptr(myws + WS_CNT + blockIdx.x * 512)) : "memory");
    Chunk c0, c1, c2, c3;
    push_load(c0, xs, svo); push_load(c1, xs, svo + 512u); push_load(c2, xs, svo + H); push_load(c3, xs, svo + H + 512u);
    asm volatile("s_waitcnt vmcnt(32)\n\tv_readfirstlane_b32 %0, %1\n\ts_nop 4" : "=s"(call) : "v"(ticket) : "memory");
    call += 1u;
    const char* dst = sptr(pws + WS_DATA + (size_t)(call & 1u) * (NR * SHARD) + (size_t)p.rank * SHARD + (size_t)kq * QUART + 4096);
    const char* flag = sptr(pws + WS_PROG + (p.rank * 4 + kq) * 128);
    const unsigned dvo = (unsigned)lane * 16u, zero = 0u, seq = call << 7;

    push_wait<24>(c0);                       push_store(c0, dst, dvo);          push_load(c0, xs, svo + IT);
    push_wait<32>(c1);                                 push_store(c1, dst, dvo + 8192u);  push_load(c1, xs, svo + IT + 512u);
    push_wait<24>(c2); push_flag(flag, zero, seq + 1); push_store(c2, dst, dvo + 16384u); push_load(c2, xs, svo + IT + H);
    push_wait<25>(c3); push_flag(flag, zero, seq + 2); push_store(c3, dst, dvo + 24576u); push_load(c3, xs, svo + IT + H + 512u);
#pragma unroll 1
    for (unsigned i = 4; i < 64; i += 4) {
        const unsigned nb = (i < 60 ? i + 4 : i) * (IT / 4);
        const unsigned d = dvo + i * 8192u;

        push_wait<25>(c0); push_flag(flag, zero, seq + i - 1); push_store(c0, dst, d);          push_load(c0, xs, svo + nb);
        push_wait<25>(c1); push_flag(flag, zero, seq + i);     push_store(c1, dst, d + 8192u);  push_load(c1, xs, svo + nb + 512u);
        push_wait<25>(c2); push_flag(flag, zero, seq + i + 1); push_store(c2, dst, d + 16384u); push_load(c2, xs, svo + nb + H);
        push_wait<25>(c3); push_flag(flag, zero, seq + i + 2); push_store(c3, dst, d + 24576u); push_load(c3, xs, svo + nb + H + 512u);
    }
    wait_vm<25>(); push_flag(flag, zero, seq + 63);
    wait_vm<9>();  push_flag(flag, zero, seq + 64);

    push_wait<0>(c0); push_wait<0>(c1); push_wait<0>(c2); push_wait<0>(c3);

}

#define PANEL_MAP 2
__global__ void __launch_bounds__(THREADS, 1) ag_gemm_kernel(Params p) {
    __shared__ __attribute__((aligned(16))) char smem[LDS_BYTES];
    const unsigned lane = threadIdx.x & 63;
    const unsigned kq = __builtin_amdgcn_readfirstlane(threadIdx.x >> 6);
    if (blockIdx.x < NPUSH) {
        if (kq == 0 && (blockIdx.x & 7) != 0) pusher(p, lane);
        return;
    }
    const unsigned label = blockIdx.x & 7, idx = (blockIdx.x - NPUSH) >> 3, rank = (unsigned)p.rank;
    const unsigned panel = (label >> 1) * 8 + (idx < 24 ? (idx & 7) : (label & 1) * 4 + (idx & 3)), rg = idx < 24 ? (label & 1) * 4 + (idx >> 3) : 3;
    const unsigned src = (rank + 1 + rg) & 7u;
    const unsigned lds0 = (unsigned)(size_t)(__attribute__((address_space(3))) char*)smem;
    const char* wb = p.w + (size_t)kq * 1024 * NL + panel * 64;
    const unsigned wvo = (lane >> 2) * NL + (((lane & 3) ^ ((lane >> 4) & 1)) * 16);
    const unsigned g = lane >> 4, q = (lane & 15) >> 1;
    const unsigned tr = lds0 + kq * 32768 + (16 * g + q) * 64 + (((q >> 2) | ((g & 1) << 1)) * 16) + (lane & 1) * 8;
    constexpr unsigned BAND = PANEL_MAP == 0 ? 256 : PANEL_MAP == 2 ? 512 : 1024, RPU = 1024 / BAND, NU = KD / RPU, NREM = NU - (NU / 112) * 112;
    const unsigned wi = idx * 4 + kq, pfc = (label * 256) & ~(BAND - 1), pfl0 = (lane / (BAND / 16)) * NL + pfc + (lane % (BAND / 16)) * 16;
    const unsigned pfo = wi * RPU * NL + pfl0, pfl = (NU - NREM + wi % NREM) * RPU * NL + pfl0;
    const unsigned u = ((blockIdx.x - NPUSH) * 4 + kq) & 511u, ub = u >> 4, uq = (u >> 2) & 3, us = u & 3;
    const unsigned rs = (ub * 16 + (lane & 15)) * KD + uq * 1024 + us * 256 + (lane >> 4) * 16;
    const unsigned rd = rank * SHARD + uq * QUART + ub * 16384 + us * 4096 + lane * 16;
    const unsigned m = lane & 15, fm = 4 * kq + g;
    const unsigned pw = lds0 + kq * 4096 + m * 256 + ((g ^ (m & 7)) * 16);
    const unsigned pr = lds0 + fm * 256 + ((m ^ (fm & 7)) * 16);
    const unsigned vout = (fm * NL + panel * 64 + m * 4) * 4;
    const unsigned nloc = rg < 2 ? 3 : 2, nitem = nloc + NITEM;
    const unsigned nb = lane < 4 ? 0 : min((lane - 4) / 3 + 1, nloc);
    const bool loc = lane >= 4 && (lane - 4) % 3 == 0 && (lane - 4) / 3 < nloc, rem = !loc && lane < nitem;
    const unsigned tb = rem ? lane - nb : (loc ? rg + 7 * (nb - 1) : 0), ts = loc ? rank : src;
    const unsigned tabx = ts * SHARD + kq * QUART + tb * 32768 + 4096, tabo = (ts * MS + tb * 32) * (NL * 4), tabn = rem ? 4 * tb : 0, isrem = rem ? ~0u : 0u;
    asm volatile(
        "v_mov_b32 v32, 0\n\t"
        "v_mov_b32 v33, 1\n\t"
        "s_load_dword s54, %[sx], 0x0\n\t"
        "s_load_dword s55, %[sw], 0x0\n\t"
        "s_mov_b64 s[40:41], %[w]\n\t"
        "s_mov_b64 s[70:71], %[ws]\n\t"
        "s_mov_b64 s[72:73], %[out]\n\t"
        "s_add_u32 s56, s70, %[cnt]\n\t"
        "s_addc_u32 s57, s71, 0\n\t"
        "s_mov_b64 exec, 1\n\t"
        "global_atomic_add v34, v32, v33, s[56:57] sc0\n\t"
        "s_mov_b64 exec, -1\n\t"
        "global_load_dwordx4 v[128:131], %[rs], %[x] offset:0\n\t"
        "global_load_dwordx4 v[132:135], %[rs], %[x] offset:64\n\t"
        "global_load_dwordx4 v[136:139], %[rs], %[x] offset:128\n\t"
        "global_load_dwordx4 v[140:143], %[rs], %[x] offset:192\n\t"
        "s_mov_b64 s[46:47], %[wfull]\n\t"
        "global_load_dwordx4 v[144:147], %[pfo], s[46:47]\n\t"
        "s_add_u32 s46, s46, 458752\n\t"
        "s_addc_u32 s47, s47, 0\n\t"
        "global_load_dwordx4 v[148:151], %[pfo], s[46:47]\n\t"
        "s_add_u32 s46, s46, 458752\n\t"
        "s_addc_u32 s47, s47, 0\n\t"
        "global_load_dwordx4 v[152:155], %[pfo], s[46:47]\n\t"
        "s_add_u32 s46, s46, 458752\n\t"
        "s_addc_u32 s47, s47, 0\n\t"
        "global_load_dwordx4 v[156:159], %[pfo], s[46:47]\n\t"
        "s_add_u32 s46, s46, 458752\n\t"
        "s_addc_u32 s47, s47, 0\n\t"
        "global_load_dwordx4 v[160:163], %[pfo], s[46:47]\n\t"
        "s_add_u32 s46, s46, 458752\n\t"
        "s_addc_u32 s47, s47, 0\n\t"
        "global_load_dwordx4 v[164:167], %[pfo], s[46:47]\n\t"
        "s_add_u32 s46, s46, 458752\n\t"
        "s_addc_u32 s47, s47, 0\n\t"
        "global_load_dwordx4 v[168:171], %[pfo], s[46:47]\n\t"
        "s_add_u32 s46, s46, 458752\n\t"
        "s_addc_u32 s47, s47, 0\n\t"
        "global_load_dwordx4 v[172:175], %[pfo], s[46:47]\n\t"
        "s_add_u32 s46, s46, 458752\n\t"
        "s_addc_u32 s47, s47, 0\n\t"
        "global_load_dwordx4 v[176:179], %[pfo], s[46:47]\n\t"
        "s_add_u32 s46, s46, 458752\n\t"
        "s_addc_u32 s47, s47, 0\n\t"
        "global_load_dwordx4 v[180:183], %[pfo], s[46:47]\n\t"
        "s_add_u32 s46, s46, 458752\n\t"
        "s_addc_u32 s47, s47, 0\n\t"
        "global_load_dwordx4 v[184:187], %[pfo], s[46:47]\n\t"
        "s_add_u32 s46, s46, 458752\n\t"
        "s_addc_u32 s47, s47, 0\n\t"
        "global_load_dwordx4 v[188:191], %[pfo], s[46:47]\n\t"
        "s_add_u32 s46, s46, 458752\n\t"
        "s_addc_u32 s47, s47, 0\n\t"
        "global_load_dwordx4 v[192:195], %[pfo], s[46:47]\n\t"
        "s_add_u32 s46, s46, 458752\n\t"
        "s_addc_u32 s47, s47, 0\n\t"
        "global_load_dwordx4 v[196:199], %[pfo], s[46:47]\n\t"
        "s_add_u32 s46, s46, 458752\n\t"
        "s_addc_u32 s47, s47, 0\n\t"
        "global_load_dwordx4 v[200:203], %[pfo], s[46:47]\n\t"
        "s_add_u32 s46, s46, 458752\n\t"
        "s_addc_u32 s47, s47, 0\n\t"
        "global_load_dwordx4 v[204:207], %[pfo], s[46:47]\n\t"
        "s_add_u32 s46, s46, 458752\n\t"
        "s_addc_u32 s47, s47, 0\n\t"
        "global_load_dwordx4 v[208:211], %[pfo], s[46:47]\n\t"
        "s_add_u32 s46, s46, 458752\n\t"
        "s_addc_u32 s47, s47, 0\n\t"
        "global_load_dwordx4 v[212:215], %[pfo], s[46:47]\n\t"
        "s_add_u32 s46, s46, 458752\n\t"
        "s_addc_u32 s47, s47, 0\n\t"
        "global_load_dwordx4 v[216:219], %[pfl], %[wfull]\n\t"
        "s_mov_b32 s74, %[ldsw]\n\t"
        "v_xor_b32 v48, 32, %[wvo]\n\t"
        "v_mov_b32 v43, %[badaddr]\n\t"
        "s_waitcnt lgkmcnt(0)\n\t"
        "v_mov_b32 v42, s54\n\t"
        "v_mul_f32 v42, s55, v42\n\t"
        "s_cmp_lg_u32 %[kq], 0\n\t"
        "s_cbranch_scc1 .Lnoinit%=\n\t"
        "ds_write_b32 v43, v42\n\t"
        ".Lnoinit%=:\n\t"
        "v_xor_b32 v45, 16, %[tr]\n\t"
        "v_xor_b32 v46, 32, %[tr]\n\t"
        "v_xor_b32 v47, 48, %[tr]\n\t"
        "v_xor_b32 v44, 64, %[pw]\n\t"
        "s_waitcnt vmcnt(19)\n\t"
        "v_readfirstlane_b32 s48, v34\n\t"
        "s_add_u32 s48, s48, 1\n\t"
        "s_lshl_b32 s49, s48, 7\n\t"
        "s_and_b32 s54, s48, 1\n\t"
        "s_lshl_b32 s54, s54, 24\n\t"
        "s_add_u32 s54, s54, 0x100000\n\t"
        "s_add_u32 s42, s70, s54\n\t"
        "s_addc_u32 s43, s71, 0\n\t"
        "global_store_dwordx4 %[rd], v[128:131], s[42:43] offset:0 sc0 sc1\n\t"
        "global_store_dwordx4 %[rd], v[132:135], s[42:43] offset:1024 sc0 sc1\n\t"
        "global_store_dwordx4 %[rd], v[136:139], s[42:43] offset:2048 sc0 sc1\n\t"
        "global_store_dwordx4 %[rd], v[140:143], s[42:43] offset:3072 sc0 sc1\n\t"
        "v_and_b32 v38, s49, %[isrem]\n\t"
        "v_add_u32 v38, v38, %[tabn]\n\t"
        "s_add_u32 s58, s70, %[tmooff]\n\t"
        "s_addc_u32 s59, s71, 0\n\t"
        "s_waitcnt vmcnt(4)\n\t"
        "s_add_u32 m0, s74, 0\n\t"
        "s_nop 0\n\t"
        "global_load_lds_dwordx4 %[wvo], s[40:41]\n\t"
        "s_add_u32 s40, s40, 0x8000\n\t"
        "s_addc_u32 s41, s41, 0\n\t"
        "s_add_u32 m0, s74, 1024\n\t"
        "s_nop 0\n\t"
        "global_load_lds_dwordx4 v48, s[40:41]\n\t"
        "s_add_u32 s40, s40, 0x8000\n\t"
        "s_addc_u32 s41, s41, 0\n\t"
        "s_add_u32 m0, s74, 2048\n\t"
        "s_nop 0\n\t"
        "global_load_lds_dwordx4 %[wvo], s[40:41]\n\t"
        "s_add_u32 s40, s40, 0x8000\n\t"
        "s_addc_u32 s41, s41, 0\n\t"
        "s_add_u32 m0, s74, 3072\n\t"
        "s_nop 0\n\t"
        "global_load_lds_dwordx4 v48, s[40:41]\n\t"
        "s_add_u32 s40, s40, 0x8000\n\t"
        "s_addc_u32 s41, s41, 0\n\t"
        "s_add_u32 m0, s74, 4096\n\t"
        "s_nop 0\n\t"
        "global_load_lds_dwordx4 %[wvo], s[40:41]\n\t"
        "s_add_u32 s40, s40, 0x8000\n\t"
        "s_addc_u32 s41, s41, 0\n\t"
        "s_add_u32 m0, s74, 5120\n\t"
        "s_nop 0\n\t"
        "global_load_lds_dwordx4 v48, s[40:41]\n\t"
        "s_add_u32 s40, s40, 0x8000\n\t"
        "s_addc_u32 s41, s41, 0\n\t"
        "s_add_u32 m0, s74, 6144\n\t"
        "s_nop 0\n\t"
        "global_load_lds_dwordx4 %[wvo], s[40:41]\n\t"
        "s_add_u32 s40, s40, 0x8000\n\t"
        "s_addc_u32 s41, s41, 0\n\t"
        "s_add_u32 m0, s74, 7168\n\t"
        "s_nop 0\n\t"
        "global_load_lds_dwordx4 v48, s[40:41]\n\t"
        "s_add_u32 s40, s40, 0x8000\n\t"
        "s_addc_u32 s41, s41, 0\n\t"
        "s_add_u32 m0, s74, 8192\n\t"
        "s_nop 0\n\t"
        "global_load_lds_dwordx4 %[wvo], s[40:41]\n\t"
        "s_add_u32 s40, s40, 0x8000\n\t"
        "s_addc_u32 s41, s41, 0\n\t"
        "s_add_u32 m0, s74, 9216\n\t"
        "s_nop 0\n\t"
        "global_load_lds_dwordx4 v48, s[40:41]\n\t"
        "s_add_u32 s40, s40, 0x8000\n\t"
        "s_addc_u32 s41, s41, 0\n\t"
        "s_add_u32 m0, s74, 10240\n\t"
        "s_nop 0\n\t"
        "global_load_lds_dwordx4 %[wvo], s[40:41]\n\t"
        "s_add_u32 s40, s40, 0x8000\n\t"
        "s_addc_u32 s41, s41, 0\n\t"
        "s_add_u32 m0, s74, 11264\n\t"
        "s_nop 0\n\t"
        "global_load_lds_dwordx4 v48, s[40:41]\n\t"
        "s_add_u32 s40, s40, 0x8000\n\t"
        "s_addc_u32 s41, s41, 0\n\t"
        "s_add_u32 m0, s74, 12288\n\t"
        "s_nop 0\n\t"
        "global_load_lds_dwordx4 %[wvo], s[40:41]\n\t"
        "s_add_u32 s40, s40, 0x8000\n\t"
        "s_addc_u32 s41, s41, 0\n\t"
        "s_add_u32 m0, s74, 13312\n\t"
        "s_nop 0\n\t"
        "global_load_lds_dwordx4 v48, s[40:41]\n\t"
        "s_add_u32 s40, s40, 0x8000\n\t"
        "s_addc_u32 s41, s41, 0\n\t"
        "s_add_u32 m0, s74, 14336\n\t"
        "s_nop 0\n\t"
        "global_load_lds_dwordx4 %[wvo], s[40:41]\n\t"
        "s_add_u32 s40, s40, 0x8000\n\t"
        "s_addc_u32 s41, s41, 0\n\t"
        "s_add_u32 m0, s74, 15360\n\t"
        "s_nop 0\n\t"
        "global_load_lds_dwordx4 v48, s[40:41]\n\t"
        "s_add_u32 s40, s40, 0x8000\n\t"
        "s_addc_u32 s41, s41, 0\n\t"
        "s_add_u32 m0, s74, 16384\n\t"
        "s_nop 0\n\t"
        "global_load_lds_dwordx4 %[wvo], s[40:41]\n\t"
        "s_add_u32 s40, s40, 0x8000\n\t"
        "s_addc_u32 s41, s41, 0\n\t"
        "s_add_u32 m0, s74, 17408\n\t"
        "s_nop 0\n\t"
        "global_load_lds_dwordx4 v48, s[40:41]\n\t"
        "s_add_u32 s40, s40, 0x8000\n\t"
        "s_addc_u32 s41, s41, 0\n\t"
        "s_add_u32 m0, s74, 18432\n\t"
        "s_nop 0\n\t"
        "global_load_lds_dwordx4 %[wvo], s[40:41]\n\t"
        "s_add_u32 s40, s40, 0x8000\n\t"
        "s_addc_u32 s41, s41, 0\n\t"
        "s_add_u32 m0, s74, 19456\n\t"
        "s_nop 0\n\t"
        "global_load_lds_dwordx4 v48, s[40:41]\n\t"
        "s_add_u32 s40, s40, 0x8000\n\t"
        "s_addc_u32 s41, s41, 0\n\t"
        "s_add_u32 m0, s74, 20480\n\t"
        "s_nop 0\n\t"
        "global_load_lds_dwordx4 %[wvo], s[40:41]\n\t"
        "s_add_u32 s40, s40, 0x8000\n\t"
        "s_addc_u32 s41, s41, 0\n\t"
        "s_add_u32 m0, s74, 21504\n\t"
        "s_nop 0\n\t"
        "global_load_lds_dwordx4 v48, s[40:41]\n\t"
        "s_add_u32 s40, s40, 0x8000\n\t"
        "s_addc_u32 s41, s41, 0\n\t"
        "s_add_u32 m0, s74, 22528\n\t"
        "s_nop 0\n\t"
        "global_load_lds_dwordx4 %[wvo], s[40:41]\n\t"
        "s_add_u32 s40, s40, 0x8000\n\t"
        "s_addc_u32 s41, s41, 0\n\t"
        "s_add_u32 m0, s74, 23552\n\t"
        "s_nop 0\n\t"
        "global_load_lds_dwordx4 v48, s[40:41]\n\t"
        "s_add_u32 s40, s40, 0x8000\n\t"
        "s_addc_u32 s41, s41, 0\n\t"
        "s_add_u32 m0, s74, 24576\n\t"
        "s_nop 0\n\t"
        "global_load_lds_dwordx4 %[wvo], s[40:41]\n\t"
        "s_add_u32 s40, s40, 0x8000\n\t"
        "s_addc_u32 s41, s41, 0\n\t"
        "s_add_u32 m0, s74, 25600\n\t"
        "s_nop 0\n\t"
        "global_load_lds_dwordx4 v48, s[40:41]\n\t"
        "s_add_u32 s40, s40, 0x8000\n\t"
        "s_addc_u32 s41, s41, 0\n\t"
        "s_add_u32 m0, s74, 26624\n\t"
        "s_nop 0\n\t"
        "global_load_lds_dwordx4 %[wvo], s[40:41]\n\t"
        "s_add_u32 s40, s40, 0x8000\n\t"
        "s_addc_u32 s41, s41, 0\n\t"
        "s_add_u32 m0, s74, 27648\n\t"
        "s_nop 0\n\t"
        "global_load_lds_dwordx4 v48, s[40:41]\n\t"
        "s_add_u32 s40, s40, 0x8000\n\t"
        "s_addc_u32 s41, s41, 0\n\t"
        "s_add_u32 m0, s74, 28672\n\t"
        "s_nop 0\n\t"
        "global_load_lds_dwordx4 %[wvo], s[40:41]\n\t"
        "s_add_u32 s40, s40, 0x8000\n\t"
        "s_addc_u32 s41, s41, 0\n\t"
        "s_add_u32 m0, s74, 29696\n\t"
        "s_nop 0\n\t"
        "global_load_lds_dwordx4 v48, s[40:41]\n\t"
        "s_add_u32 s40, s40, 0x8000\n\t"
        "s_addc_u32 s41, s41, 0\n\t"
        "s_add_u32 m0, s74, 30720\n\t"
        "s_nop 0\n\t"
        "global_load_lds_dwordx4 %[wvo], s[40:41]\n\t"
        "s_add_u32 s40, s40, 0x8000\n\t"
        "s_addc_u32 s41, s41, 0\n\t"
        "s_add_u32 m0, s74, 31744\n\t"
        "s_nop 0\n\t"
        "global_load_lds_dwordx4 v48, s[40:41]\n\t"
        "s_add_u32 s40, s40, 0x8000\n\t"
        "s_addc_u32 s41, s41, 0\n\t"
        "s_waitcnt vmcnt(32) lgkmcnt(0)\n\t"
        "s_barrier\n\t"
        "s_add_u32 s56, s70, %[rdyoff]\n\t"
        "s_addc_u32 s57, s71, 0\n\t"
        "s_cmp_lg_u32 %[kq], 0\n\t"
        "s_cbranch_scc1 .Lnoadd%=\n\t"
        "s_mov_b64 exec, 1\n\t"
        "global_atomic_add v32, v33, s[56:57]\n\t"
        "s_mov_b64 exec, -1\n\t"
        ".Lnoadd%=:\n\t"
        "s_waitcnt vmcnt(28)\n\t"
        "ds_read_b64_tr_b8 a[0:1], %[tr] offset:0\n\t"
        "ds_read_b64_tr_b8 a[2:3], %[tr] offset:512\n\t"
        "ds_read_b64_tr_b8 a[4:5], v45 offset:0\n\t"
        "ds_read_b64_tr_b8 a[6:7], v45 offset:512\n\t"
        "ds_read_b64_tr_b8 a[8:9], v46 offset:0\n\t"
        "ds_read_b64_tr_b8 a[10:11], v46 offset:512\n\t"
        "ds_read_b64_tr_b8 a[12:13], v47 offset:0\n\t"
        "ds_read_b64_tr_b8 a[14:15], v47 offset:512\n\t"
        "s_waitcnt lgkmcnt(0)\n\t"
        "s_add_u32 m0, s74, 0\n\t"
        "s_nop 0\n\t"
        "global_load_lds_dwordx4 %[wvo], s[40:41]\n\t"
        "s_add_u32 s40, s40, 0x8000\n\t"
        "s_addc_u32 s41, s41, 0\n\t"
        "s_add_u32 m0, s74, 1024\n\t"
        "s_nop 0\n\t"
        "global_load_lds_dwordx4 v48, s[40:41]\n\t"
        "s_add_u32 s40, s40, 0x8000\n\t"
        "s_addc_u32 s41, s41, 0\n\t"
        "s_add_u32 m0, s74, 2048\n\t"
        "s_nop 0\n\t"
        "global_load_lds_dwordx4 %[wvo], s[40:41]\n\t"
        "s_add_u32 s40, s40, 0x8000\n\t"
        "s_addc_u32 s41, s41, 0\n\t"
        "s_add_u32 m0, s74, 3072\n\t"
        "s_nop 0\n\t"
        "global_load_lds_dwordx4 v48, s[40:41]\n\t"
        "s_add_u32 s40, s40, 0x8000\n\t"
        "s_addc_u32 s41, s41, 0\n\t"
        "s_waitcnt vmcnt(28)\n\t"
        "ds_read_b64_tr_b8 a[16:17], %[tr] offset:4096\n\t"
        "ds_read_b64_tr_b8 a[18:19], %[tr] offset:4608\n\t"
        "ds_read_b64_tr_b8 a[20:21], v45 offset:4096\n\t"
        "ds_read_b64_tr_b8 a[22:23], v45 offset:4608\n\t"
        "ds_read_b64_tr_b8 a[24:25], v46 offset:4096\n\t"
        "ds_read_b64_tr_b8 a[26:27], v46 offset:4608\n\t"
        "ds_read_b64_tr_b8 a[28:29], v47 offset:4096\n\t"
        "ds_read_b64_tr_b8 a[30:31], v47 offset:4608\n\t"
        "s_waitcnt lgkmcnt(0)\n\t"
        "s_add_u32 m0, s74, 4096\n\t"
        "s_nop 0\n\t"
        "global_load_lds_dwordx4 %[wvo], s[40:41]\n\t"
        "s_add_u32 s40, s40, 0x8000\n\t"
        "s_addc_u32 s41, s41, 0\n\t"
        "s_add_u32 m0, s74, 5120\n\t"
        "s_nop 0\n\t"
        "global_load_lds_dwordx4 v48, s[40:41]\n\t"
        "s_add_u32 s40, s40, 0x8000\n\t"
        "s_addc_u32 s41, s41, 0\n\t"
        "s_add_u32 m0, s74, 6144\n\t"
        "s_nop 0\n\t"
        "global_load_lds_dwordx4 %[wvo], s[40:41]\n\t"
        "s_add_u32 s40, s40, 0x8000\n\t"
        "s_addc_u32 s41, s41, 0\n\t"
        "s_add_u32 m0, s74, 7168\n\t"
        "s_nop 0\n\t"
        "global_load_lds_dwordx4 v48, s[40:41]\n\t"
        "s_add_u32 s40, s40, 0x8000\n\t"
        "s_addc_u32 s41, s41, 0\n\t"
        "s_waitcnt vmcnt(28)\n\t"
        "ds_read_b64_tr_b8 a[32:33], %[tr] offset:8192\n\t"
        "ds_read_b64_tr_b8 a[34:35], %[tr] offset:8704\n\t"
        "ds_read_b64_tr_b8 a[36:37], v45 offset:8192\n\t"
        "ds_read_b64_tr_b8 a[38:39], v45 offset:8704\n\t"
        "ds_read_b64_tr_b8 a[40:41], v46 offset:8192\n\t"
        "ds_read_b64_tr_b8 a[42:43], v46 offset:8704\n\t"
        "ds_read_b64_tr_b8 a[44:45], v47 offset:8192\n\t"
        "ds_read_b64_tr_b8 a[46:47], v47 offset:8704\n\t"
        "s_waitcnt lgkmcnt(0)\n\t"
        "s_add_u32 m0, s74, 8192\n\t"
        "s_nop 0\n\t"
        "global_load_lds_dwordx4 %[wvo], s[40:41]\n\t"
        "s_add_u32 s40, s40, 0x8000\n\t"
        "s_addc_u32 s41, s41, 0\n\t"
        "s_add_u32 m0, s74, 9216\n\t"
        "s_nop 0\n\t"
        "global_load_lds_dwordx4 v48, s[40:41]\n\t"
        "s_add_u32 s40, s40, 0x8000\n\t"
        "s_addc_u32 s41, s41, 0\n\t"
        "s_add_u32 m0, s74, 10240\n\t"
        "s_nop 0\n\t"
        "global_load_lds_dwordx4 %[wvo], s[40:41]\n\t"
        "s_add_u32 s40, s40, 0x8000\n\t"
        "s_addc_u32 s41, s41, 0\n\t"
        "s_add_u32 m0, s74, 11264\n\t"
        "s_nop 0\n\t"
        "global_load_lds_dwordx4 v48, s[40:41]\n\t"
        "s_add_u32 s40, s40, 0x8000\n\t"
        "s_addc_u32 s41, s41, 0\n\t"
        "s_waitcnt vmcnt(28)\n\t"
        "ds_read_b64_tr_b8 a[48:49], %[tr] offset:12288\n\t"
        "ds_read_b64_tr_b8 a[50:51], %[tr] offset:12800\n\t"
        "ds_read_b64_tr_b8 a[52:53], v45 offset:12288\n\t"
        "ds_read_b64_tr_b8 a[54:55], v45 offset:12800\n\t"
        "ds_read_b64_tr_b8 a[56:57], v46 offset:12288\n\t"
        "ds_read_b64_tr_b8 a[58:59], v46 offset:12800\n\t"
        "ds_read_b64_tr_b8 a[60:61], v47 offset:12288\n\t"
        "ds_read_b64_tr_b8 a[62:63], v47 offset:12800\n\t"
        "s_waitcnt lgkmcnt(0)\n\t"
        "s_add_u32 m0, s74, 12288\n\t"
        "s_nop 0\n\t"
        "global_load_lds_dwordx4 %[wvo], s[40:41]\n\t"
        "s_add_u32 s40, s40, 0x8000\n\t"
        "s_addc_u32 s41, s41, 0\n\t"
        "s_add_u32 m0, s74, 13312\n\t"
        "s_nop 0\n\t"
        "global_load_lds_dwordx4 v48, s[40:41]\n\t"
        "s_add_u32 s40, s40, 0x8000\n\t"
        "s_addc_u32 s41, s41, 0\n\t"
        "s_add_u32 m0, s74, 14336\n\t"
        "s_nop 0\n\t"
        "global_load_lds_dwordx4 %[wvo], s[40:41]\n\t"
        "s_add_u32 s40, s40, 0x8000\n\t"
        "s_addc_u32 s41, s41, 0\n\t"
        "s_add_u32 m0, s74, 15360\n\t"
        "s_nop 0\n\t"
        "global_load_lds_dwordx4 v48, s[40:41]\n\t"
        "s_add_u32 s40, s40, 0x8000\n\t"
        "s_addc_u32 s41, s41, 0\n\t"
        "s_waitcnt vmcnt(28)\n\t"
        "ds_read_b64_tr_b8 a[64:65], %[tr] offset:16384\n\t"
        "ds_read_b64_tr_b8 a[66:67], %[tr] offset:16896\n\t"
        "ds_read_b64_tr_b8 a[68:69], v45 offset:16384\n\t"
        "ds_read_b64_tr_b8 a[70:71], v45 offset:16896\n\t"
        "ds_read_b64_tr_b8 a[72:73], v46 offset:16384\n\t"
        "ds_read_b64_tr_b8 a[74:75], v46 offset:16896\n\t"
        "ds_read_b64_tr_b8 a[76:77], v47 offset:16384\n\t"
        "ds_read_b64_tr_b8 a[78:79], v47 offset:16896\n\t"
        "s_waitcnt lgkmcnt(0)\n\t"
        "s_add_u32 m0, s74, 16384\n\t"
        "s_nop 0\n\t"
        "global_load_lds_dwordx4 %[wvo], s[40:41]\n\t"
        "s_add_u32 s40, s40, 0x8000\n\t"
        "s_addc_u32 s41, s41, 0\n\t"
        "s_add_u32 m0, s74, 17408\n\t"
        "s_nop 0\n\t"
        "global_load_lds_dwordx4 v48, s[40:41]\n\t"
        "s_add_u32 s40, s40, 0x8000\n\t"
        "s_addc_u32 s41, s41, 0\n\t"
        "s_add_u32 m0, s74, 18432\n\t"
        "s_nop 0\n\t"
        "global_load_lds_dwordx4 %[wvo], s[40:41]\n\t"
        "s_add_u32 s40, s40, 0x8000\n\t"
        "s_addc_u32 s41, s41, 0\n\t"
        "s_add_u32 m0, s74, 19456\n\t"
        "s_nop 0\n\t"
        "global_load_lds_dwordx4 v48, s[40:41]\n\t"
        "s_add_u32 s40, s40, 0x8000\n\t"
        "s_addc_u32 s41, s41, 0\n\t"
        "s_waitcnt vmcnt(28)\n\t"
        "ds_read_b64_tr_b8 a[80:81], %[tr] offset:20480\n\t"
        "ds_read_b64_tr_b8 a[82:83], %[tr] offset:20992\n\t"
        "ds_read_b64_tr_b8 a[84:85], v45 offset:20480\n\t"
        "ds_read_b64_tr_b8 a[86:87], v45 offset:20992\n\t"
        "ds_read_b64_tr_b8 a[88:89], v46 offset:20480\n\t"
        "ds_read_b64_tr_b8 a[90:91], v46 offset:20992\n\t"
        "ds_read_b64_tr_b8 a[92:93], v47 offset:20480\n\t"
        "ds_read_b64_tr_b8 a[94:95], v47 offset:20992\n\t"
        "s_waitcnt lgkmcnt(0)\n\t"
        "s_add_u32 m0, s74, 20480\n\t"
        "s_nop 0\n\t"
        "global_load_lds_dwordx4 %[wvo], s[40:41]\n\t"
        "s_add_u32 s40, s40, 0x8000\n\t"
        "s_addc_u32 s41, s41, 0\n\t"
        "s_add_u32 m0, s74, 21504\n\t"
        "s_nop 0\n\t"
        "global_load_lds_dwordx4 v48, s[40:41]\n\t"
        "s_add_u32 s40, s40, 0x8000\n\t"
        "s_addc_u32 s41, s41, 0\n\t"
        "s_add_u32 m0, s74, 22528\n\t"
        "s_nop 0\n\t"
        "global_load_lds_dwordx4 %[wvo], s[40:41]\n\t"
        "s_add_u32 s40, s40, 0x8000\n\t"
        "s_addc_u32 s41, s41, 0\n\t"
        "s_add_u32 m0, s74, 23552\n\t"
        "s_nop 0\n\t"
        "global_load_lds_dwordx4 v48, s[40:41]\n\t"
        "s_add_u32 s40, s40, 0x8000\n\t"
        "s_addc_u32 s41, s41, 0\n\t"
        "s_waitcnt vmcnt(28)\n\t"
        "ds_read_b64_tr_b8 a[96:97], %[tr] offset:24576\n\t"
        "ds_read_b64_tr_b8 a[98:99], %[tr] offset:25088\n\t"
        "ds_read_b64_tr_b8 a[100:101], v45 offset:24576\n\t"
        "ds_read_b64_tr_b8 a[102:103], v45 offset:25088\n\t"
        "ds_read_b64_tr_b8 a[104:105], v46 offset:24576\n\t"
        "ds_read_b64_tr_b8 a[106:107], v46 offset:25088\n\t"
        "ds_read_b64_tr_b8 a[108:109], v47 offset:24576\n\t"
        "ds_read_b64_tr_b8 a[110:111], v47 offset:25088\n\t"
        "s_waitcnt lgkmcnt(0)\n\t"
        "s_add_u32 m0, s74, 24576\n\t"
        "s_nop 0\n\t"
        "global_load_lds_dwordx4 %[wvo], s[40:41]\n\t"
        "s_add_u32 s40, s40, 0x8000\n\t"
        "s_addc_u32 s41, s41, 0\n\t"
        "s_add_u32 m0, s74, 25600\n\t"
        "s_nop 0\n\t"
        "global_load_lds_dwordx4 v48, s[40:41]\n\t"
        "s_add_u32 s40, s40, 0x8000\n\t"
        "s_addc_u32 s41, s41, 0\n\t"
        "s_add_u32 m0, s74, 26624\n\t"
        "s_nop 0\n\t"
        "global_load_lds_dwordx4 %[wvo], s[40:41]\n\t"
        "s_add_u32 s40, s40, 0x8000\n\t"
        "s_addc_u32 s41, s41, 0\n\t"
        "s_add_u32 m0, s74, 27648\n\t"
        "s_nop 0\n\t"
        "global_load_lds_dwordx4 v48, s[40:41]\n\t"
        "s_add_u32 s40, s40, 0x8000\n\t"
        "s_addc_u32 s41, s41, 0\n\t"
        "s_waitcnt vmcnt(28)\n\t"
        "ds_read_b64_tr_b8 a[112:113], %[tr] offset:28672\n\t"
        "ds_read_b64_tr_b8 a[114:115], %[tr] offset:29184\n\t"
        "ds_read_b64_tr_b8 a[116:117], v45 offset:28672\n\t"
        "ds_read_b64_tr_b8 a[118:119], v45 offset:29184\n\t"
        "ds_read_b64_tr_b8 a[120:121], v46 offset:28672\n\t"
        "ds_read_b64_tr_b8 a[122:123], v46 offset:29184\n\t"
        "ds_read_b64_tr_b8 a[124:125], v47 offset:28672\n\t"
        "ds_read_b64_tr_b8 a[126:127], v47 offset:29184\n\t"
        "s_waitcnt lgkmcnt(0)\n\t"
        "s_add_u32 m0, s74, 28672\n\t"
        "s_nop 0\n\t"
        "global_load_lds_dwordx4 %[wvo], s[40:41]\n\t"
        "s_add_u32 s40, s40, 0x8000\n\t"
        "s_addc_u32 s41, s41, 0\n\t"
        "s_add_u32 m0, s74, 29696\n\t"
        "s_nop 0\n\t"
        "global_load_lds_dwordx4 v48, s[40:41]\n\t"
        "s_add_u32 s40, s40, 0x8000\n\t"
        "s_addc_u32 s41, s41, 0\n\t"
        "s_add_u32 m0, s74, 30720\n\t"
        "s_nop 0\n\t"
        "global_load_lds_dwordx4 %[wvo], s[40:41]\n\t"
        "s_add_u32 s40, s40, 0x8000\n\t"
        "s_addc_u32 s41, s41, 0\n\t"
        "s_add_u32 m0, s74, 31744\n\t"
        "s_nop 0\n\t"
        "global_load_lds_dwordx4 v48, s[40:41]\n\t"
        "s_add_u32 s40, s40, 0x8000\n\t"
        "s_addc_u32 s41, s41, 0\n\t"
        "s_waitcnt vmcnt(28)\n\t"
        "ds_read_b64_tr_b8 a[128:129], %[tr] offset:0\n\t"
        "ds_read_b64_tr_b8 a[130:131], %[tr] offset:512\n\t"
        "ds_read_b64_tr_b8 a[132:133], v45 offset:0\n\t"
        "ds_read_b64_tr_b8 a[134:135], v45 offset:512\n\t"
        "ds_read_b64_tr_b8 a[136:137], v46 offset:0\n\t"
        "ds_read_b64_tr_b8 a[138:139], v46 offset:512\n\t"
        "ds_read_b64_tr_b8 a[140:141], v47 offset:0\n\t"
        "ds_read_b64_tr_b8 a[142:143], v47 offset:512\n\t"
        "s_waitcnt vmcnt(24)\n\t"
        "ds_read_b64_tr_b8 a[144:145], %[tr] offset:4096\n\t"
        "ds_read_b64_tr_b8 a[146:147], %[tr] offset:4608\n\t"
        "ds_read_b64_tr_b8 a[148:149], v45 offset:4096\n\t"
        "ds_read_b64_tr_b8 a[150:151], v45 offset:4608\n\t"
        "ds_read_b64_tr_b8 a[152:153], v46 offset:4096\n\t"
        "ds_read_b64_tr_b8 a[154:155], v46 offset:4608\n\t"
        "ds_read_b64_tr_b8 a[156:157], v47 offset:4096\n\t"
        "ds_read_b64_tr_b8 a[158:159], v47 offset:4608\n\t"
        "s_waitcnt vmcnt(20)\n\t"
        "ds_read_b64_tr_b8 a[160:161], %[tr] offset:8192\n\t"
        "ds_read_b64_tr_b8 a[162:163], %[tr] offset:8704\n\t"
        "ds_read_b64_tr_b8 a[164:165], v45 offset:8192\n\t"
        "ds_read_b64_tr_b8 a[166:167], v45 offset:8704\n\t"
        "ds_read_b64_tr_b8 a[168:169], v46 offset:8192\n\t"
        "ds_read_b64_tr_b8 a[170:171], v46 offset:8704\n\t"
        "ds_read_b64_tr_b8 a[172:173], v47 offset:8192\n\t"
        "ds_read_b64_tr_b8 a[174:175], v47 offset:8704\n\t"
        "s_waitcnt vmcnt(16)\n\t"
        "ds_read_b64_tr_b8 a[176:177], %[tr] offset:12288\n\t"
        "ds_read_b64_tr_b8 a[178:179], %[tr] offset:12800\n\t"
        "ds_read_b64_tr_b8 a[180:181], v45 offset:12288\n\t"
        "ds_read_b64_tr_b8 a[182:183], v45 offset:12800\n\t"
        "ds_read_b64_tr_b8 a[184:185], v46 offset:12288\n\t"
        "ds_read_b64_tr_b8 a[186:187], v46 offset:12800\n\t"
        "ds_read_b64_tr_b8 a[188:189], v47 offset:12288\n\t"
        "ds_read_b64_tr_b8 a[190:191], v47 offset:12800\n\t"
        "s_waitcnt vmcnt(12)\n\t"
        "ds_read_b64_tr_b8 a[192:193], %[tr] offset:16384\n\t"
        "ds_read_b64_tr_b8 a[194:195], %[tr] offset:16896\n\t"
        "ds_read_b64_tr_b8 a[196:197], v45 offset:16384\n\t"
        "ds_read_b64_tr_b8 a[198:199], v45 offset:16896\n\t"
        "ds_read_b64_tr_b8 a[200:201], v46 offset:16384\n\t"
        "ds_read_b64_tr_b8 a[202:203], v46 offset:16896\n\t"
        "ds_read_b64_tr_b8 a[204:205], v47 offset:16384\n\t"
        "ds_read_b64_tr_b8 a[206:207], v47 offset:16896\n\t"
        "s_waitcnt vmcnt(8)\n\t"
        "ds_read_b64_tr_b8 a[208:209], %[tr] offset:20480\n\t"
        "ds_read_b64_tr_b8 a[210:211], %[tr] offset:20992\n\t"
        "ds_read_b64_tr_b8 a[212:213], v45 offset:20480\n\t"
        "ds_read_b64_tr_b8 a[214:215], v45 offset:20992\n\t"
        "ds_read_b64_tr_b8 a[216:217], v46 offset:20480\n\t"
        "ds_read_b64_tr_b8 a[218:219], v46 offset:20992\n\t"
        "ds_read_b64_tr_b8 a[220:221], v47 offset:20480\n\t"
        "ds_read_b64_tr_b8 a[222:223], v47 offset:20992\n\t"
        "s_waitcnt vmcnt(4)\n\t"
        "ds_read_b64_tr_b8 a[224:225], %[tr] offset:24576\n\t"
        "ds_read_b64_tr_b8 a[226:227], %[tr] offset:25088\n\t"
        "ds_read_b64_tr_b8 a[228:229], v45 offset:24576\n\t"
        "ds_read_b64_tr_b8 a[230:231], v45 offset:25088\n\t"
        "ds_read_b64_tr_b8 a[232:233], v46 offset:24576\n\t"
        "ds_read_b64_tr_b8 a[234:235], v46 offset:25088\n\t"
        "ds_read_b64_tr_b8 a[236:237], v47 offset:24576\n\t"
        "ds_read_b64_tr_b8 a[238:239], v47 offset:25088\n\t"
        "s_waitcnt vmcnt(0)\n\t"
        "ds_read_b64_tr_b8 a[240:241], %[tr] offset:28672\n\t"
        "ds_read_b64_tr_b8 a[242:243], %[tr] offset:29184\n\t"
        "ds_read_b64_tr_b8 a[244:245], v45 offset:28672\n\t"
        "ds_read_b64_tr_b8 a[246:247], v45 offset:29184\n\t"
        "ds_read_b64_tr_b8 a[248:249], v46 offset:28672\n\t"
        "ds_read_b64_tr_b8 a[250:251], v46 offset:29184\n\t"
        "ds_read_b64_tr_b8 a[252:253], v47 offset:28672\n\t"
        "ds_read_b64_tr_b8 a[254:255], v47 offset:29184\n\t"
        "s_waitcnt lgkmcnt(0)\n\t"
        "s_barrier\n\t"
        "s_mov_b32 s50, 16\n\t"
        "s_add_u32 s56, s70, %[progoff]\n\t"
        "s_addc_u32 s57, s71, 0\n\t"
        "s_mov_b32 s53, 0\n\t"
        "s_mov_b64 s[66:67], 0\n\t"
        "v_readlane_b32 s69, %[tabx], 0\n\t"
        "v_readlane_b32 s52, v38, 0\n\t"
        "v_mov_b32 v35, 0\n\t"
        "s_add_u32 s44, s42, s69\n\t"
        "s_addc_u32 s45, s43, 0\n\t"
        "s_add_u32 s51, s52, 2\n\t"
        "s_call_b64 s[60:61], .Lslow%=\n\t"
        "global_load_dwordx4 v[128:131], %[l16], s[44:45] offset:-4096 sc0 sc1\n\t"
        "global_load_dwordx4 v[132:135], %[l16], s[44:45] offset:-3072 sc0 sc1\n\t"
        "global_load_dwordx4 v[136:139], %[l16], s[44:45] offset:-2048 sc0 sc1\n\t"
        "global_load_dwordx4 v[140:143], %[l16], s[44:45] offset:-1024 sc0 sc1\n\t"
        "global_load_dwordx4 v[144:147], %[l16], s[44:45] offset:0 sc0 sc1\n\t"
        "global_load_dwordx4 v[148:151], %[l16], s[44:45] offset:1024 sc0 sc1\n\t"
        "global_load_dwordx4 v[152:155], %[l16], s[44:45] offset:2048 sc0 sc1\n\t"
        "global_load_dwordx4 v[156:159], %[l16], s[44:45] offset:3072 sc0 sc1\n\t"
        "s_add_u32 s44, s44, 0x2000\n\t"
        "s_addc_u32 s45, s45, 0\n\t"
        "global_load_dwordx4 v[160:163], %[l16], s[44:45] offset:-4096 sc0 sc1\n\t"
        "global_load_dwordx4 v[164:167], %[l16], s[44:45] offset:-3072 sc0 sc1\n\t"
        "global_load_dwordx4 v[168:171], %[l16], s[44:45] offset:-2048 sc0 sc1\n\t"
        "global_load_dwordx4 v[172:175], %[l16], s[44:45] offset:-1024 sc0 sc1\n\t"
        ".Lloop%=:\n\t"
        "s_waitcnt vmcnt(8)\n\t"
        "v_mfma_i32_16x16x64_i8 v[96:99], a[0:3], v[128:131], 0\n\t"
        "v_mfma_i32_16x16x64_i8 v[96:99], a[16:19], v[132:135], v[96:99]\n\t"
        "v_mfma_i32_16x16x64_i8 v[96:99], a[32:35], v[136:139], v[96:99]\n\t"
        "v_mfma_i32_16x16x64_i8 v[96:99], a[48:51], v[140:143], v[96:99]\n\t"
        "global_load_dwordx4 v[176:179], %[l16], s[44:45] offset:0 sc0 sc1\n\t"
        "v_mfma_i32_16x16x64_i8 v[100:103], a[4:7], v[128:131], 0\n\t"
        "v_mfma_i32_16x16x64_i8 v[100:103], a[20:23], v[132:135], v[100:103]\n\t"
        "v_mfma_i32_16x16x64_i8 v[100:103], a[36:39], v[136:139], v[100:103]\n\t"
        "v_mfma_i32_16x16x64_i8 v[100:103], a[52:55], v[140:143], v[100:103]\n\t"
        "global_load_dwordx4 v[180:183], %[l16], s[44:45] offset:1024 sc0 sc1\n\t"
        "v_mfma_i32_16x16x64_i8 v[104:107], a[8:11], v[128:131], 0\n\t"
        "ds_write_b128 %[pw], v[112:115] offset:16384\n\t"
        "v_mfma_i32_16x16x64_i8 v[104:107], a[24:27], v[132:135], v[104:107]\n\t"
        "ds_write_b128 v44, v[116:119] offset:16384\n\t"
        "v_mfma_i32_16x16x64_i8 v[104:107], a[40:43], v[136:139], v[104:107]\n\t"
        "ds_write_b128 %[pw], v[120:123] offset:16512\n\t"
        "v_mfma_i32_16x16x64_i8 v[104:107], a[56:59], v[140:143], v[104:107]\n\t"
        "global_load_dwordx4 v[184:187], %[l16], s[44:45] offset:2048 sc0 sc1\n\t"
        "v_mfma_i32_16x16x64_i8 v[108:111], a[12:15], v[128:131], 0\n\t"
        "ds_write_b128 v44, v[124:127] offset:16512\n\t"
        "v_mfma_i32_16x16x64_i8 v[108:111], a[28:31], v[132:135], v[108:111]\n\t"
        "v_mfma_i32_16x16x64_i8 v[108:111], a[44:47], v[136:139], v[108:111]\n\t"
        "v_mfma_i32_16x16x64_i8 v[108:111], a[60:63], v[140:143], v[108:111]\n\t"
        "global_load_dwordx4 v[188:191], %[l16], s[44:45] offset:3072 sc0 sc1\n\t"
        "s_waitcnt vmcnt(8)\n\t"
        "v_mfma_i32_16x16x64_i8 v[96:99], a[64:67], v[144:147], v[96:99]\n\t"
        "v_readfirstlane_b32 s75, v35\n\t"
        "s_add_u32 s51, s52, 3\n\t"
        "v_mfma_i32_16x16x64_i8 v[96:99], a[80:83], v[148:151], v[96:99]\n\t"
        "s_add_u32 s44, s44, 0x2000\n\t"
        "s_addc_u32 s45, s45, 0\n\t"
        "s_max_u32 s50, s50, s75\n\t"
        "v_mfma_i32_16x16x64_i8 v[96:99], a[96:99], v[152:155], v[96:99]\n\t"
        "s_add_u32 s75, s51, 1\n\t"
        "s_cmp_ge_u32 s50, s75\n\t"
        "s_cbranch_scc1 .Lgoc2%=\n\t"
        "global_load_dword v35, v32, s[56:57] sc0 sc1\n\t"
        "s_cmp_ge_u32 s50, s51\n\t"
        "s_cbranch_scc1 .Lgoc2%=\n\t"
        "s_call_b64 s[60:61], .Lslow%=\n\t"
        ".Lgoc2%=:\n\t"
        "v_mfma_i32_16x16x64_i8 v[96:99], a[112:115], v[156:159], v[96:99]\n\t"
        "global_load_dwordx4 v[192:195], %[l16], s[44:45] offset:-4096 sc0 sc1\n\t"
        "s_cmp_eq_u32 s53, %[jw]\n\t"
        "s_cbranch_scc1 .Lrdy%=\n\t"
        ".Lrdyback%=:\n\t"
        "v_mfma_i32_16x16x64_i8 v[100:103], a[68:71], v[144:147], v[100:103]\n\t"
        "s_waitcnt lgkmcnt(0)\n\t"
        "s_barrier\n\t"
        "v_mfma_i32_16x16x64_i8 v[100:103], a[84:87], v[148:151], v[100:103]\n\t"
        "ds_read_b128 v[56:59], %[pr] offset:16384\n\t"
        "ds_read_b128 v[60:63], %[pr] offset:20480\n\t"
        "v_mfma_i32_16x16x64_i8 v[100:103], a[100:103], v[152:155], v[100:103]\n\t"
        "ds_read_b128 v[64:67], %[pr] offset:24576\n\t"
        "ds_read_b128 v[68:71], %[pr] offset:28672\n\t"
        "v_mfma_i32_16x16x64_i8 v[100:103], a[116:119], v[156:159], v[100:103]\n\t"
        "global_load_dwordx4 v[196:199], %[l16], s[44:45] offset:-3072 sc0 sc1\n\t"
        "ds_read_b32 v42, v43\n\t"
        "v_mfma_i32_16x16x64_i8 v[104:107], a[72:75], v[144:147], v[104:107]\n\t"
        "s_sub_u32 s55, s53, 1\n\t"
        "v_readlane_b32 s55, %[tabo], s55\n\t"
        "v_mfma_i32_16x16x64_i8 v[104:107], a[88:91], v[148:151], v[104:107]\n\t"
        "s_add_u32 s55, s55, 0x20000\n\t"
        "s_add_u32 s46, s72, s55\n\t"
        "s_addc_u32 s47, s73, 0\n\t"
        "v_mfma_i32_16x16x64_i8 v[104:107], a[104:107], v[152:155], v[104:107]\n\t"
        "v_mfma_i32_16x16x64_i8 v[104:107], a[120:123], v[156:159], v[104:107]\n\t"
        "global_load_dwordx4 v[200:203], %[l16], s[44:45] offset:-2048 sc0 sc1\n\t"
        "v_mfma_i32_16x16x64_i8 v[108:111], a[76:79], v[144:147], v[108:111]\n\t"
        "v_mfma_i32_16x16x64_i8 v[108:111], a[92:95], v[148:151], v[108:111]\n\t"
        "v_mfma_i32_16x16x64_i8 v[108:111], a[108:111], v[152:155], v[108:111]\n\t"
        "v_mfma_i32_16x16x64_i8 v[108:111], a[124:127], v[156:159], v[108:111]\n\t"
        "global_load_dwordx4 v[204:207], %[l16], s[44:45] offset:-1024 sc0 sc1\n\t"
        "s_waitcnt vmcnt(8)\n\t"
        "v_mfma_i32_16x16x64_i8 v[96:99], a[128:131], v[160:163], v[96:99]\n\t"
        "v_mfma_i32_16x16x64_i8 v[96:99], a[144:147], v[164:167], v[96:99]\n\t"
        "v_mfma_i32_16x16x64_i8 v[96:99], a[160:163], v[168:171], v[96:99]\n\t"
        "v_mfma_i32_16x16x64_i8 v[96:99], a[176:179], v[172:175], v[96:99]\n\t"
        "global_load_dwordx4 v[208:211], %[l16], s[44:45] offset:0 sc0 sc1\n\t"
        "v_mfma_i32_16x16x64_i8 v[100:103], a[132:135], v[160:163], v[100:103]\n\t"
        "s_waitcnt lgkmcnt(0)\n\t"
        "v_add_u32 v56, v56, v60\n\t"
        "v_mfma_i32_16x16x64_i8 v[100:103], a[148:151], v[164:167], v[100:103]\n\t"
        "v_add_u32 v57, v57, v61\n\t"
        "v_add_u32 v58, v58, v62\n\t"
        "v_mfma_i32_16x16x64_i8 v[100:103], a[164:167], v[168:171], v[100:103]\n\t"
        "v_add_u32 v59, v59, v63\n\t"
        "v_add3_u32 v56, v56, v64, v68\n\t"
        "v_mfma_i32_16x16x64_i8 v[100:103], a[180:183], v[172:175], v[100:103]\n\t"
        "global_load_dwordx4 v[212:215], %[l16], s[44:45] offset:1024 sc0 sc1\n\t"
        "v_add3_u32 v57, v57, v65, v69\n\t"
        "v_mfma_i32_16x16x64_i8 v[104:107], a[136:139], v[160:163], v[104:107]\n\t"
        "v_add3_u32 v58, v58, v66, v70\n\t"
        "v_add3_u32 v59, v59, v67, v71\n\t"
        "v_mfma_i32_16x16x64_i8 v[104:107], a[152:155], v[164:167], v[104:107]\n\t"
        "v_cvt_f32_i32 v56, v56\n\t"
        "v_cvt_f32_i32 v57, v57\n\t"
        "v_mfma_i32_16x16x64_i8 v[104:107], a[168:171], v[168:171], v[104:107]\n\t"
        "v_cvt_f32_i32 v58, v58\n\t"
        "v_cvt_f32_i32 v59, v59\n\t"
        "v_mfma_i32_16x16x64_i8 v[104:107], a[184:187], v[172:175], v[104:107]\n\t"
        "global_load_dwordx4 v[216:219], %[l16], s[44:45] offset:2048 sc0 sc1\n\t"
        "v_mul_f32 v56, v42, v56\n\t"
        "v_mfma_i32_16x16x64_i8 v[108:111], a[140:143], v[160:163], v[108:111]\n\t"
        "v_mul_f32 v57, v42, v57\n\t"
        "v_mul_f32 v58, v42, v58\n\t"
        "v_mfma_i32_16x16x64_i8 v[108:111], a[156:159], v[164:167], v[108:111]\n\t"
        "v_mul_f32 v59, v42, v59\n\t"
        "v_mfma_i32_16x16x64_i8 v[108:111], a[172:175], v[168:171], v[108:111]\n\t"
        "s_mov_b64 exec, s[66:67]\n\t"
        "global_store_dwordx4 %[vout], v[56:59], s[46:47]\n\t"
        "s_mov_b64 exec, -1\n\t"
        "s_mov_b64 s[66:67], -1\n\t"
        "v_mfma_i32_16x16x64_i8 v[108:111], a[188:191], v[172:175], v[108:111]\n\t"
        "global_load_dwordx4 v[220:223], %[l16], s[44:45] offset:3072 sc0 sc1\n\t"
        "s_waitcnt vmcnt(8)\n\t"
        "v_mfma_i32_16x16x64_i8 v[96:99], a[192:195], v[176:179], v[96:99]\n\t"
        "v_readfirstlane_b32 s75, v35\n\t"
        "s_add_u32 s51, s52, 4\n\t"
        "v_mfma_i32_16x16x64_i8 v[96:99], a[208:211], v[180:183], v[96:99]\n\t"
        "s_add_u32 s44, s44, 0x2000\n\t"
        "s_addc_u32 s45, s45, 0\n\t"
        "s_max_u32 s50, s50, s75\n\t"
        "v_mfma_i32_16x16x64_i8 v[96:99], a[224:227], v[184:187], v[96:99]\n\t"
        "s_add_u32 s75, s51, 1\n\t"
        "s_cmp_ge_u32 s50, s75\n\t"
        "s_cbranch_scc1 .Lgoc3%=\n\t"
        "global_load_dword v35, v32, s[56:57] sc0 sc1\n\t"
        "s_cmp_ge_u32 s50, s51\n\t"
        "s_cbranch_scc1 .Lgoc3%=\n\t"
        "s_call_b64 s[60:61], .Lslow%=\n\t"
        ".Lgoc3%=:\n\t"
        "v_mfma_i32_16x16x64_i8 v[96:99], a[240:243], v[188:191], v[96:99]\n\t"
        "global_load_dwordx4 v[224:227], %[l16], s[44:45] offset:-4096 sc0 sc1\n\t"
        "v_mfma_i32_16x16x64_i8 v[100:103], a[196:199], v[176:179], v[100:103]\n\t"
        "v_mfma_i32_16x16x64_i8 v[100:103], a[212:215], v[180:183], v[100:103]\n\t"
        "v_mfma_i32_16x16x64_i8 v[100:103], a[228:231], v[184:187], v[100:103]\n\t"
        "v_mfma_i32_16x16x64_i8 v[100:103], a[244:247], v[188:191], v[100:103]\n\t"
        "global_load_dwordx4 v[228:231], %[l16], s[44:45] offset:-3072 sc0 sc1\n\t"
        "v_mfma_i32_16x16x64_i8 v[104:107], a[200:203], v[176:179], v[104:107]\n\t"
        "v_mfma_i32_16x16x64_i8 v[104:107], a[216:219], v[180:183], v[104:107]\n\t"
        "v_mfma_i32_16x16x64_i8 v[104:107], a[232:235], v[184:187], v[104:107]\n\t"
        "v_mfma_i32_16x16x64_i8 v[104:107], a[248:251], v[188:191], v[104:107]\n\t"
        "global_load_dwordx4 v[232:235], %[l16], s[44:45] offset:-2048 sc0 sc1\n\t"
        "v_mfma_i32_16x16x64_i8 v[108:111], a[204:207], v[176:179], v[108:111]\n\t"
        "v_mfma_i32_16x16x64_i8 v[108:111], a[220:223], v[180:183], v[108:111]\n\t"
        "v_mfma_i32_16x16x64_i8 v[108:111], a[236:239], v[184:187], v[108:111]\n\t"
        "v_mfma_i32_16x16x64_i8 v[108:111], a[252:255], v[188:191], v[108:111]\n\t"
        "global_load_dwordx4 v[236:239], %[l16], s[44:45] offset:-1024 sc0 sc1\n\t"
        "s_waitcnt vmcnt(8)\n\t"
        "v_mfma_i32_16x16x64_i8 v[112:115], a[0:3], v[192:195], 0\n\t"
        "v_mfma_i32_16x16x64_i8 v[112:115], a[16:19], v[196:199], v[112:115]\n\t"
        "v_mfma_i32_16x16x64_i8 v[112:115], a[32:35], v[200:203], v[112:115]\n\t"
        "v_mfma_i32_16x16x64_i8 v[112:115], a[48:51], v[204:207], v[112:115]\n\t"
        "global_load_dwordx4 v[240:243], %[l16], s[44:45] offset:0 sc0 sc1\n\t"
        "v_mfma_i32_16x16x64_i8 v[116:119], a[4:7], v[192:195], 0\n\t"
        "s_add_u32 s54, s53, 1\n\t"
        "v_readlane_b32 s69, %[tabx], s54\n\t"
        "v_mfma_i32_16x16x64_i8 v[116:119], a[20:23], v[196:199], v[116:119]\n\t"
        "v_readlane_b32 s52, v38, s54\n\t"
        "v_mfma_i32_16x16x64_i8 v[116:119], a[36:39], v[200:203], v[116:119]\n\t"
        "v_mfma_i32_16x16x64_i8 v[116:119], a[52:55], v[204:207], v[116:119]\n\t"
        "global_load_dwordx4 v[244:247], %[l16], s[44:45] offset:1024 sc0 sc1\n\t"
        "v_mfma_i32_16x16x64_i8 v[120:123], a[8:11], v[192:195], 0\n\t"
        "ds_write_b128 %[pw], v[96:99] offset:0\n\t"
        "v_mfma_i32_16x16x64_i8 v[120:123], a[24:27], v[196:199], v[120:123]\n\t"
        "ds_write_b128 v44, v[100:103] offset:0\n\t"
        "v_mfma_i32_16x16x64_i8 v[120:123], a[40:43], v[200:203], v[120:123]\n\t"
        "ds_write_b128 %[pw], v[104:107] offset:128\n\t"
        "v_mfma_i32_16x16x64_i8 v[120:123], a[56:59], v[204:207], v[120:123]\n\t"
        "global_load_dwordx4 v[248:251], %[l16], s[44:45] offset:2048 sc0 sc1\n\t"
        "v_mfma_i32_16x16x64_i8 v[124:127], a[12:15], v[192:195], 0\n\t"
        "ds_write_b128 v44, v[108:111] offset:128\n\t"
        "v_mfma_i32_16x16x64_i8 v[124:127], a[28:31], v[196:199], v[124:127]\n\t"
        "v_mfma_i32_16x16x64_i8 v[124:127], a[44:47], v[200:203], v[124:127]\n\t"
        "v_mfma_i32_16x16x64_i8 v[124:127], a[60:63], v[204:207], v[124:127]\n\t"
        "global_load_dwordx4 v[252:255], %[l16], s[44:45] offset:3072 sc0 sc1\n\t"
        "s_waitcnt vmcnt(8)\n\t"
        "v_mfma_i32_16x16x64_i8 v[112:115], a[64:67], v[208:211], v[112:115]\n\t"
        "v_readfirstlane_b32 s75, v35\n\t"
        "s_add_u32 s51, s52, 1\n\t"
        "v_mfma_i32_16x16x64_i8 v[112:115], a[80:83], v[212:215], v[112:115]\n\t"
        "s_add_u32 s44, s42, s69\n\t"
        "s_addc_u32 s45, s43, 0\n\t"
        "s_max_u32 s50, s50, s75\n\t"
        "v_mfma_i32_16x16x64_i8 v[112:115], a[96:99], v[216:219], v[112:115]\n\t"
        "s_add_u32 s75, s51, 1\n\t"
        "s_cmp_ge_u32 s50, s75\n\t"
        "s_cbranch_scc1 .Lgoc0%=\n\t"
        "global_load_dword v35, v32, s[56:57] sc0 sc1\n\t"
        "s_cmp_ge_u32 s50, s51\n\t"
        "s_cbranch_scc1 .Lgoc0%=\n\t"
        "s_call_b64 s[60:61], .Lslow%=\n\t"
        ".Lgoc0%=:\n\t"
        "v_mfma_i32_16x16x64_i8 v[112:115], a[112:115], v[220:223], v[112:115]\n\t"
        "global_load_dwordx4 v[128:131], %[l16], s[44:45] offset:-4096 sc0 sc1\n\t"
        "v_mfma_i32_16x16x64_i8 v[116:119], a[68:71], v[208:211], v[116:119]\n\t"
        "s_waitcnt lgkmcnt(0)\n\t"
        "s_barrier\n\t"
        "v_mfma_i32_16x16x64_i8 v[116:119], a[84:87], v[212:215], v[116:119]\n\t"
        "ds_read_b128 v[56:59], %[pr] offset:0\n\t"
        "ds_read_b128 v[60:63], %[pr] offset:4096\n\t"
        "v_mfma_i32_16x16x64_i8 v[116:119], a[100:103], v[216:219], v[116:119]\n\t"
        "ds_read_b128 v[64:67], %[pr] offset:8192\n\t"
        "ds_read_b128 v[68:71], %[pr] offset:12288\n\t"
        "v_mfma_i32_16x16x64_i8 v[116:119], a[116:119], v[220:223], v[116:119]\n\t"
        "global_load_dwordx4 v[132:135], %[l16], s[44:45] offset:-3072 sc0 sc1\n\t"
        "ds_read_b32 v42, v43\n\t"
        "v_mfma_i32_16x16x64_i8 v[120:123], a[72:75], v[208:211], v[120:123]\n\t"
        "v_readlane_b32 s55, %[tabo], s53\n\t"
        "s_add_u32 s46, s72, s55\n\t"
        "s_addc_u32 s47, s73, 0\n\t"
        "v_mfma_i32_16x16x64_i8 v[120:123], a[88:91], v[212:215], v[120:123]\n\t"
        "v_mfma_i32_16x16x64_i8 v[120:123], a[104:107], v[216:219], v[120:123]\n\t"
        "v_mfma_i32_16x16x64_i8 v[120:123], a[120:123], v[220:223], v[120:123]\n\t"
        "global_load_dwordx4 v[136:139], %[l16], s[44:45] offset:-2048 sc0 sc1\n\t"
        "v_mfma_i32_16x16x64_i8 v[124:127], a[76:79], v[208:211], v[124:127]\n\t"
        "v_mfma_i32_16x16x64_i8 v[124:127], a[92:95], v[212:215], v[124:127]\n\t"
        "v_mfma_i32_16x16x64_i8 v[124:127], a[108:111], v[216:219], v[124:127]\n\t"
        "v_mfma_i32_16x16x64_i8 v[124:127], a[124:127], v[220:223], v[124:127]\n\t"
        "global_load_dwordx4 v[140:143], %[l16], s[44:45] offset:-1024 sc0 sc1\n\t"
        "s_waitcnt vmcnt(8)\n\t"
        "v_mfma_i32_16x16x64_i8 v[112:115], a[128:131], v[224:227], v[112:115]\n\t"
        "v_mfma_i32_16x16x64_i8 v[112:115], a[144:147], v[228:231], v[112:115]\n\t"
        "v_mfma_i32_16x16x64_i8 v[112:115], a[160:163], v[232:235], v[112:115]\n\t"
        "v_mfma_i32_16x16x64_i8 v[112:115], a[176:179], v[236:239], v[112:115]\n\t"
        "global_load_dwordx4 v[144:147], %[l16], s[44:45] offset:0 sc0 sc1\n\t"
        "v_mfma_i32_16x16x64_i8 v[116:119], a[132:135], v[224:227], v[116:119]\n\t"
        "s_waitcnt lgkmcnt(0)\n\t"
        "v_add_u32 v56, v56, v60\n\t"
        "v_mfma_i32_16x16x64_i8 v[116:119], a[148:151], v[228:231], v[116:119]\n\t"
        "v_add_u32 v57, v57, v61\n\t"
        "v_add_u32 v58, v58, v62\n\t"
        "v_mfma_i32_16x16x64_i8 v[116:119], a[164:167], v[232:235], v[116:119]\n\t"
        "v_add_u32 v59, v59, v63\n\t"
        "v_add3_u32 v56, v56, v64, v68\n\t"
        "v_mfma_i32_16x16x64_i8 v[116:119], a[180:183], v[236:239], v[116:119]\n\t"
        "global_load_dwordx4 v[148:151], %[l16], s[44:45] offset:1024 sc0 sc1\n\t"
        "v_add3_u32 v57, v57, v65, v69\n\t"
        "v_mfma_i32_16x16x64_i8 v[120:123], a[136:139], v[224:227], v[120:123]\n\t"
        "v_add3_u32 v58, v58, v66, v70\n\t"
        "v_add3_u32 v59, v59, v67, v71\n\t"
        "v_mfma_i32_16x16x64_i8 v[120:123], a[152:155], v[228:231], v[120:123]\n\t"
        "v_cvt_f32_i32 v56, v56\n\t"
        "v_cvt_f32_i32 v57, v57\n\t"
        "v_mfma_i32_16x16x64_i8 v[120:123], a[168:171], v[232:235], v[120:123]\n\t"
        "v_cvt_f32_i32 v58, v58\n\t"
        "v_cvt_f32_i32 v59, v59\n\t"
        "v_mfma_i32_16x16x64_i8 v[120:123], a[184:187], v[236:239], v[120:123]\n\t"
        "global_load_dwordx4 v[152:155], %[l16], s[44:45] offset:2048 sc0 sc1\n\t"
        "v_mul_f32 v56, v42, v56\n\t"
        "v_mfma_i32_16x16x64_i8 v[124:127], a[140:143], v[224:227], v[124:127]\n\t"
        "v_mul_f32 v57, v42, v57\n\t"
        "v_mul_f32 v58, v42, v58\n\t"
        "v_mfma_i32_16x16x64_i8 v[124:127], a[156:159], v[228:231], v[124:127]\n\t"
        "v_mul_f32 v59, v42, v59\n\t"
        "v_mfma_i32_16x16x64_i8 v[124:127], a[172:175], v[232:235], v[124:127]\n\t"
        "global_store_dwordx4 %[vout], v[56:59], s[46:47]\n\t"
        "v_mfma_i32_16x16x64_i8 v[124:127], a[188:191], v[236:239], v[124:127]\n\t"
        "global_load_dwordx4 v[156:159], %[l16], s[44:45] offset:3072 sc0 sc1\n\t"
        "s_waitcnt vmcnt(8)\n\t"
        "v_mfma_i32_16x16x64_i8 v[112:115], a[192:195], v[240:243], v[112:115]\n\t"
        "v_readfirstlane_b32 s75, v35\n\t"
        "s_add_u32 s51, s52, 2\n\t"
        "v_mfma_i32_16x16x64_i8 v[112:115], a[208:211], v[244:247], v[112:115]\n\t"
        "s_add_u32 s44, s44, 0x2000\n\t"
        "s_addc_u32 s45, s45, 0\n\t"
        "s_max_u32 s50, s50, s75\n\t"
        "v_mfma_i32_16x16x64_i8 v[112:115], a[224:227], v[248:251], v[112:115]\n\t"
        "s_add_u32 s75, s51, 1\n\t"
        "s_cmp_ge_u32 s50, s75\n\t"
        "s_cbranch_scc1 .Lgoc1%=\n\t"
        "global_load_dword v35, v32, s[56:57] sc0 sc1\n\t"
        "s_cmp_ge_u32 s50, s51\n\t"
        "s_cbranch_scc1 .Lgoc1%=\n\t"
        "s_call_b64 s[60:61], .Lslow%=\n\t"
        ".Lgoc1%=:\n\t"
        "v_mfma_i32_16x16x64_i8 v[112:115], a[240:243], v[252:255], v[112:115]\n\t"
        "global_load_dwordx4 v[160:163], %[l16], s[44:45] offset:-4096 sc0 sc1\n\t"
        "v_mfma_i32_16x16x64_i8 v[116:119], a[196:199], v[240:243], v[116:119]\n\t"
        "v_mfma_i32_16x16x64_i8 v[116:119], a[212:215], v[244:247], v[116:119]\n\t"
        "v_mfma_i32_16x16x64_i8 v[116:119], a[228:231], v[248:251], v[116:119]\n\t"
        "v_mfma_i32_16x16x64_i8 v[116:119], a[244:247], v[252:255], v[116:119]\n\t"
        "global_load_dwordx4 v[164:167], %[l16], s[44:45] offset:-3072 sc0 sc1\n\t"
        "v_mfma_i32_16x16x64_i8 v[120:123], a[200:203], v[240:243], v[120:123]\n\t"
        "v_mfma_i32_16x16x64_i8 v[120:123], a[216:219], v[244:247], v[120:123]\n\t"
        "v_mfma_i32_16x16x64_i8 v[120:123], a[232:235], v[248:251], v[120:123]\n\t"
        "v_mfma_i32_16x16x64_i8 v[120:123], a[248:251], v[252:255], v[120:123]\n\t"
        "global_load_dwordx4 v[168:171], %[l16], s[44:45] offset:-2048 sc0 sc1\n\t"
        "v_mfma_i32_16x16x64_i8 v[124:127], a[204:207], v[240:243], v[124:127]\n\t"
        "s_add_u32 s53, s53, 1\n\t"
        "s_cmp_lt_u32 s53, %[nitem]\n\t"
        "v_mfma_i32_16x16x64_i8 v[124:127], a[220:223], v[244:247], v[124:127]\n\t"
        "v_mfma_i32_16x16x64_i8 v[124:127], a[236:239], v[248:251], v[124:127]\n\t"
        "v_mfma_i32_16x16x64_i8 v[124:127], a[252:255], v[252:255], v[124:127]\n\t"
        "global_load_dwordx4 v[172:175], %[l16], s[44:45] offset:-1024 sc0 sc1\n\t"
        "s_cbranch_scc1 .Lloop%=\n\t"
        "s_sub_u32 s55, s53, 1\n\t"
        "v_readlane_b32 s55, %[tabo], s55\n\t"
        "s_add_u32 s55, s55, 0x20000\n\t"
        "s_add_u32 s46, s72, s55\n\t"
        "s_addc_u32 s47, s73, 0\n\t"
        "s_nop 2\n\t"
        "ds_write_b128 %[pw], v[112:115] offset:16384\n\t"
        "ds_write_b128 v44, v[116:119] offset:16384\n\t"
        "ds_write_b128 %[pw], v[120:123] offset:16512\n\t"
        "ds_write_b128 v44, v[124:127] offset:16512\n\t"
        "s_waitcnt lgkmcnt(0)\n\t"
        "s_barrier\n\t"
        "ds_read_b128 v[56:59], %[pr] offset:16384\n\t"
        "ds_read_b128 v[60:63], %[pr] offset:20480\n\t"
        "ds_read_b128 v[64:67], %[pr] offset:24576\n\t"
        "ds_read_b128 v[68:71], %[pr] offset:28672\n\t"
        "ds_read_b32 v42, v43\n\t"
        "s_waitcnt lgkmcnt(0)\n\t"
        "v_add_u32 v56, v56, v60\n\t"
        "v_add_u32 v57, v57, v61\n\t"
        "v_add_u32 v58, v58, v62\n\t"
        "v_add_u32 v59, v59, v63\n\t"
        "v_add3_u32 v56, v56, v64, v68\n\t"
        "v_add3_u32 v57, v57, v65, v69\n\t"
        "v_add3_u32 v58, v58, v66, v70\n\t"
        "v_add3_u32 v59, v59, v67, v71\n\t"
        "v_cvt_f32_i32 v56, v56\n\t"
        "v_cvt_f32_i32 v57, v57\n\t"
        "v_cvt_f32_i32 v58, v58\n\t"
        "v_cvt_f32_i32 v59, v59\n\t"
        "v_mul_f32 v56, v42, v56\n\t"
        "v_mul_f32 v57, v42, v57\n\t"
        "v_mul_f32 v58, v42, v58\n\t"
        "v_mul_f32 v59, v42, v59\n\t"
        "global_store_dwordx4 %[vout], v[56:59], s[46:47]\n\t"
        "s_waitcnt vmcnt(0)\n\t"
        "s_branch .Lend%=\n\t"
        ".Lrdy%=:\n\t"
        "s_mov_b64 s[76:77], s[56:57]\n\t"
        "s_mov_b32 s78, s50\n\t"
        "s_add_u32 s56, s70, %[rdyoff]\n\t"
        "s_addc_u32 s57, s71, 0\n\t"
        "s_mul_i32 s51, s48, 224\n\t"
        "s_cmp_eq_u32 s50, -1\n\t"
        "s_cselect_b32 s50, -1, 0\n\t"
        "s_call_b64 s[60:61], .Lslow%=\n\t"
        "s_cmp_eq_u32 s50, -1\n\t"
        "s_cselect_b32 s50, -1, s78\n\t"
        "s_mov_b64 s[56:57], s[76:77]\n\t"
        "s_branch .Lrdyback%=\n\t"
        ".Lslow%=:\n\t"
        "s_mov_b32 s65, 0\n\t"
        "s_mov_b32 s64, 0\n\t"
        ".Lspin%=:\n\t"
        "global_load_dword v37, v32, s[56:57] sc0 sc1\n\t"
        "s_add_u32 s65, s65, 1\n\t"
        "s_waitcnt vmcnt(0)\n\t"
        "v_readfirstlane_b32 s75, v37\n\t"
        "s_max_u32 s50, s50, s75\n\t"
        "s_cmp_ge_u32 s50, s51\n\t"
        "s_cbranch_scc1 .Lback%=\n\t"
        "s_and_b32 s75, s65, 31\n\t"
        "s_cbranch_scc1 .Lspin%=\n\t"
        "s_memrealtime s[62:63]\n\t"
        "global_load_dword v37, v32, s[58:59] sc1\n\t"
        "s_waitcnt vmcnt(0) lgkmcnt(0)\n\t"
        "v_readfirstlane_b32 s75, v37\n\t"
        "s_cmp_eq_u32 s75, s48\n\t"
        "s_cbranch_scc1 .Ldead%=\n\t"
        "s_cmp_eq_u32 s64, 0\n\t"
        "s_cselect_b32 s64, s62, s64\n\t"
        "s_sub_u32 s75, s62, s64\n\t"
        "s_sleep 8\n\t"
        "s_cmp_lt_u32 s75, %[deadline]\n\t"
        "s_cbranch_scc1 .Lspin%=\n\t"
        ".Ldead%=:\n\t"
        "v_mov_b32 v37, s48\n\t"
        "global_store_dword v32, v37, s[58:59] sc1\n\t"
        "v_mov_b32 v33, 0x7fc00000\n\t"
        "ds_write_b32 v43, v33\n\t"
        "s_mov_b32 s50, -1\n\t"
        "s_waitcnt vmcnt(0) lgkmcnt(0)\n\t"
        ".Lback%=:\n\t"
        "s_setpc_b64 s[60:61]\n\t"
        ".Lend%=:\n\t"
        :
        : [w] "s"(wb), [ws] "s"(p.ws[0]), [out] "s"(p.out), [x] "s"(p.x), [wfull] "s"(p.w), [kq] "s"(kq), [jw] "s"(kq == 0 ? 2u : ~0u), [nitem] "s"(nitem), [sx] "s"(p.sx), [sw] "s"(p.sw),
          [cnt] "s"(WS_CNT + (blockIdx.x * 4 + kq) * 128), [progoff] "s"(WS_PROG + (src * 4 + kq) * 128), [ldsw] "s"(lds0 + kq * 32768),
          [badaddr] "s"(lds0 + LDS_SCALE), [tmooff] "i"(WS_TMO), [rdyoff] "i"(WS_RDY), [deadline] "i"(DEADLINE_TICKS),
          [wvo] "v"(wvo), [pfo] "v"(pfo), [pfl] "v"(pfl), [tr] "v"(tr), [rs] "v"(rs), [rd] "v"(rd), [l16] "v"(lane * 16), [pw] "v"(pw), [pr] "v"(pr), [vout] "v"(vout),
          [tabx] "v"(tabx), [tabo] "v"(tabo), [tabn] "v"(tabn), [isrem] "v"(isrem)
        :
          "v32", "v33", "v34", "v35", "v36", "v37", "v38", "v39", "v40", "v41", "v42", "v43", "v44", "v45", "v46", "v47", "v48", "v49", "v50", "v51", "v52", "v53", "v54", "v55",
          "v56", "v57", "v58", "v59", "v60", "v61", "v62", "v63", "v64", "v65", "v66", "v67", "v68", "v69", "v70", "v71", "v72", "v73", "v74", "v75", "v76", "v77", "v78", "v79",
          "v80", "v81", "v82", "v83", "v84", "v85", "v86", "v87", "v88", "v89", "v90", "v91", "v92", "v93", "v94", "v95", "v96", "v97", "v98", "v99", "v100", "v101", "v102", "v103",
          "v104", "v105", "v106", "v107", "v108", "v109", "v110", "v111", "v112", "v113", "v114", "v115", "v116", "v117", "v118", "v119", "v120", "v121", "v122", "v123", "v124", "v125", "v126", "v127",
          "v128", "v129", "v130", "v131", "v132", "v133", "v134", "v135", "v136", "v137", "v138", "v139", "v140", "v141", "v142", "v143", "v144", "v145", "v146", "v147", "v148", "v149", "v150", "v151",
          "v152", "v153", "v154", "v155", "v156", "v157", "v158", "v159", "v160", "v161", "v162", "v163", "v164", "v165", "v166", "v167", "v168", "v169", "v170", "v171", "v172", "v173", "v174", "v175",
          "v176", "v177", "v178", "v179", "v180", "v181", "v182", "v183", "v184", "v185", "v186", "v187", "v188", "v189", "v190", "v191", "v192", "v193", "v194", "v195", "v196", "v197", "v198", "v199",
          "v200", "v201", "v202", "v203", "v204", "v205", "v206", "v207", "v208", "v209", "v210", "v211", "v212", "v213", "v214", "v215", "v216", "v217", "v218", "v219", "v220", "v221", "v222", "v223",
          "v224", "v225", "v226", "v227", "v228", "v229", "v230", "v231", "v232", "v233", "v234", "v235", "v236", "v237", "v238", "v239", "v240", "v241", "v242", "v243", "v244", "v245", "v246", "v247",
          "v248", "v249", "v250", "v251", "v252", "v253", "v254", "v255", "a0", "a1", "a2", "a3", "a4", "a5", "a6", "a7", "a8", "a9", "a10", "a11", "a12", "a13", "a14", "a15",
          "a16", "a17", "a18", "a19", "a20", "a21", "a22", "a23", "a24", "a25", "a26", "a27", "a28", "a29", "a30", "a31", "a32", "a33", "a34", "a35", "a36", "a37", "a38", "a39",
          "a40", "a41", "a42", "a43", "a44", "a45", "a46", "a47", "a48", "a49", "a50", "a51", "a52", "a53", "a54", "a55", "a56", "a57", "a58", "a59", "a60", "a61", "a62", "a63",
          "a64", "a65", "a66", "a67", "a68", "a69", "a70", "a71", "a72", "a73", "a74", "a75", "a76", "a77", "a78", "a79", "a80", "a81", "a82", "a83", "a84", "a85", "a86", "a87",
          "a88", "a89", "a90", "a91", "a92", "a93", "a94", "a95", "a96", "a97", "a98", "a99", "a100", "a101", "a102", "a103", "a104", "a105", "a106", "a107", "a108", "a109", "a110", "a111",
          "a112", "a113", "a114", "a115", "a116", "a117", "a118", "a119", "a120", "a121", "a122", "a123", "a124", "a125", "a126", "a127", "a128", "a129", "a130", "a131", "a132", "a133", "a134", "a135",
          "a136", "a137", "a138", "a139", "a140", "a141", "a142", "a143", "a144", "a145", "a146", "a147", "a148", "a149", "a150", "a151", "a152", "a153", "a154", "a155", "a156", "a157", "a158", "a159",
          "a160", "a161", "a162", "a163", "a164", "a165", "a166", "a167", "a168", "a169", "a170", "a171", "a172", "a173", "a174", "a175", "a176", "a177", "a178", "a179", "a180", "a181", "a182", "a183",
          "a184", "a185", "a186", "a187", "a188", "a189", "a190", "a191", "a192", "a193", "a194", "a195", "a196", "a197", "a198", "a199", "a200", "a201", "a202", "a203", "a204", "a205", "a206", "a207",
          "a208", "a209", "a210", "a211", "a212", "a213", "a214", "a215", "a216", "a217", "a218", "a219", "a220", "a221", "a222", "a223", "a224", "a225", "a226", "a227", "a228", "a229", "a230", "a231",
          "a232", "a233", "a234", "a235", "a236", "a237", "a238", "a239", "a240", "a241", "a242", "a243", "a244", "a245", "a246", "a247", "a248", "a249", "a250", "a251", "a252", "a253", "a254", "a255",
          "s40", "s41", "s42", "s43", "s44", "s45", "s46", "s47", "s48", "s49", "s50", "s51", "s52", "s53", "s54", "s55", "s56", "s57", "s58", "s59", "s60", "s61", "s62", "s63",
          "s64", "s65", "s66", "s67", "s68", "s69", "s70", "s71", "s72", "s73", "s74", "s75", "s76", "s77", "s78", "s79", "vcc", "scc", "memory");
}

extern "C" void kernel_launch(int rank, int nranks, ncclComm_t comm, hipStream_t stream,
                              void* const* d_in, const int* in_sizes, int n_in,
                              void* d_out, int out_size, const int* in_dtypes, int out_dtype,
                              void* const* d_out_all, void* const* d_ws_all, size_t ws_bytes) {
    if (nranks != NR || n_in != 4 || ws_bytes < WS_NEED || in_sizes[0] != (int)SHARD || in_sizes[1] != KD * NL || out_size != NR * MS * NL) {
        fprintf(stderr, "kernel_launch: unexpected problem shape\n");
        return;
    }
    Params p;
    p.x = (const char*)d_in[0]; p.w = (const char*)d_in[1]; p.sx = (const float*)d_in[2]; p.sw = (const float*)d_in[3];
    p.out = (float*)d_out;
    for (int r = 0; r < NR; ++r) p.ws[r] = (char*)d_ws_all[(rank + r) % NR];
    p.rank = rank; p.pad = 0;
    hipLaunchKernelGGL(ag_gemm_kernel, dim3(GRID), dim3(THREADS), 0, stream, p);
    HIP_CHECK(hipGetLastError());
}
```
